```python
import numpy as np

kernel_name = "gpu_gemm_rs_m4096_k4096_n16384_f32_silu"


ABSMAX_RTOL = 0.001

COMPUTE_DTYPE = "float32"
EPILOGUE = "silu"
IN_QUANT = None
OUT_QUANT = None
IN_DTYPES = {"x": "float32", "w_mat": "float32"}
WORLD_SIZE = 8

def setup_inputs():

    x = np.random.randn(4096, 4096).astype(np.float32)
    w_mat = (np.random.randn(4096, 16384) / np.sqrt(4096)).astype(np.float32)
    return {"x": x, "w_mat": w_mat}

def _epilogue(y):
    return (y / (1.0 + np.exp(-np.clip(y, -60.0, 60.0)))).astype(np.float32)

def reference(x, w_mat):
    y = (x @ w_mat).astype(np.float32)
    return _epilogue(y)
```

```cpp
#include <hip/hip_runtime.h>
#include <rccl/rccl.h>
#include <cstdio>
#include <cstdint>

typedef _Float16 half8 __attribute__((ext_vector_type(8)));
typedef float f32x4 __attribute__((ext_vector_type(4)));

namespace {

constexpr int NR = 8;
constexpr int MTOT = 4096;
constexpr int KC = 512;
constexpr int NTOT = 16384;
constexpr int NLOC = NTOT / NR;
constexpr int MLOC = MTOT / NR;
constexpr int NU = 8;
constexpr int NKS = NR * NU;
constexpr int NSL = NKS * 2;

constexpr size_t CTL_CNT = 0;
constexpr size_t CTL_DONE = 0x100;
constexpr size_t CTL_TICKET = 0x200;
constexpr size_t CTL_TMO = 0x300;
constexpr size_t STG_BASE = 1u << 20;
constexpr size_t XA_SLICE = 256u << 10;
constexpr size_t WB_SLICE = 128u << 10;
constexpr size_t XA_BYTES = XA_SLICE * NSL;
constexpr size_t WB_BYTES = WB_SLICE * NSL;
constexpr size_t STG_BYTES = XA_BYTES + WB_BYTES;

constexpr int NPUSH = 256;
constexpr int NGEMM = 256;
constexpr int NWG = NPUSH + NGEMM;
constexpr int NTHR = 512;

constexpr int W_ITEMS = 64, X_ITEMS = 32, R_ITEMS = W_ITEMS + X_ITEMS;
constexpr int ITEMS = R_ITEMS * NU;
constexpr unsigned CONTRIB = 8 + 32;
constexpr unsigned DONE_CONTRIB = 32 * (NR - 1);

constexpr unsigned long long DEADLINE_TICKS = 50000000ull;

constexpr int SLOT_A = 16384, SLOT_B = 8192, SLOT = SLOT_A + SLOT_B;
constexpr int NSLOT = 3;
constexpr int LDS_MISC = NSLOT * SLOT;
constexpr int LDS_BYTES = LDS_MISC + 64;

struct Tabs {
  char* ws[NR];
  float* out[NR];
};

__device__ __forceinline__ unsigned ld_sys(const unsigned* p) {
  return __hip_atomic_load(p, __ATOMIC_RELAXED, __HIP_MEMORY_SCOPE_SYSTEM);
}

__device__ __forceinline__ half8 cvt8(const float4& a, const float4& b) {
  half8 h;
  h[0] = (_Float16)a.x; h[1] = (_Float16)a.y; h[2] = (_Float16)a.z; h[3] = (_Float16)a.w;
  h[4] = (_Float16)b.x; h[5] = (_Float16)b.y; h[6] = (_Float16)b.z; h[7] = (_Float16)b.w;
  return h;
}

__device__ __forceinline__ float silu(float y) {
  float c = fminf(fmaxf(y, -60.0f), 60.0f);
  return y / (1.0f + __expf(-c));
}

}

__global__ __launch_bounds__(NTHR, 4) void fused_gemm_rs(const float* __restrict__ x,
                                                         const float* __restrict__ w,
                                                         Tabs tabs, int rank) {
  extern __shared__ __attribute__((aligned(16))) char lds[];
  const int tid = threadIdx.x;
  const int lane = tid & 63;
  const int wv = tid >> 6;
  char* myws = tabs.ws[rank];
  volatile unsigned* misc = (volatile unsigned*)(lds + LDS_MISC);

  if (tid == 0) {
    unsigned t = __hip_atomic_fetch_add((unsigned*)(myws + CTL_TICKET), 1u, __ATOMIC_RELAXED,
                                        __HIP_MEMORY_SCOPE_AGENT);
    misc[0] = t / NWG + 1u;
    misc[1] = 0u;
  }
  __syncthreads();
  const unsigned call = misc[0];
  const size_t stg = STG_BASE + (size_t)(call & 1u) * STG_BYTES;

  if ((int)blockIdx.x < NPUSH) {
    for (int it = blockIdx.x; it < ITEMS; it += NPUSH) {
      const int u = it / R_ITEMS, q = it % R_ITEMS;
      const int ks = rank * NU + u;
      if (q < W_ITEMS) {
        const int p = (q + rank) & 7, nb = q >> 3;
        const int nq = tid & 63, kg = tid >> 6;
        const float* src = w + (size_t)(64 * u + 8 * kg) * NTOT + (NLOC * p + 256 * nb + 4 * nq);
        float4 v[8];
#pragma unroll
        for (int j = 0; j < 8; ++j) v[j] = *(const float4*)(src + (size_t)j * NTOT);
        const int kh = kg >> 2, kq = kg & 3;
        const int nl = 4 * nq;
        char* dl = lds + (((kh * 16 + (nl >> 4)) * 64 + kq * 16 + (nl & 15)) << 4);
        half8 h;
#pragma unroll
        for (int j = 0; j < 8; ++j) h[j] = (_Float16)v[j].x;
        *(half8*)(dl) = h;
#pragma unroll
        for (int j = 0; j < 8; ++j) h[j] = (_Float16)v[j].y;
        *(half8*)(dl + 16) = h;
#pragma unroll
        for (int j = 0; j < 8; ++j) h[j] = (_Float16)v[j].z;
        *(half8*)(dl + 32) = h;
#pragma unroll
        for (int j = 0; j < 8; ++j) h[j] = (_Float16)v[j].w;
        *(half8*)(dl + 48) = h;
        __syncthreads();
        char* dst = tabs.ws[p] + stg + XA_BYTES + (size_t)(ks * 2) * WB_SLICE + (size_t)nb * 16384;
#pragma unroll
        for (int i = 0; i < 4; ++i) {
          const int off = (i * NTHR + tid) << 4;
          const half8 hv = *(const half8*)(lds + off);
          *(half8*)(dst + (size_t)(off >> 14) * WB_SLICE + (off & 16383)) = hv;
        }
        asm volatile("s_waitcnt vmcnt(0)" ::: "memory");
        __syncthreads();
        if (tid == 0) {
          __builtin_amdgcn_fence(__ATOMIC_RELEASE, "");
          asm volatile("s_waitcnt vmcnt(0)" ::: "memory");
          __hip_atomic_fetch_add((unsigned*)(tabs.ws[p] + CTL_CNT) + ks, 1u, __ATOMIC_RELAXED,
                                 __HIP_MEMORY_SCOPE_SYSTEM);
        }
      } else {
        const int rb = q - W_ITEMS;
        const int rg = 8 * rb + wv;
        const float* src = x + (size_t)(16 * rg + (lane & 15)) * KC + 64 * u + 8 * (lane >> 4);
        const float4 a0 = *(const float4*)(src), a1 = *(const float4*)(src + 4);
        const float4 b0 = *(const float4*)(src + 32), b1 = *(const float4*)(src + 36);
        const half8 h0 = cvt8(a0, a1), h1 = cvt8(b0, b1);
        const size_t off = stg + (size_t)(ks * 2) * XA_SLICE + (size_t)rg * 1024 + lane * 16;
#pragma unroll
        for (int d = 1; d <= NR; ++d) {
          char* dst = tabs.ws[(rank + d) & 7] + off;
          *(half8*)(dst) = h0;
          *(half8*)(dst + XA_SLICE) = h1;
        }
        asm volatile("s_waitcnt vmcnt(0)" ::: "memory");
        __syncthreads();
        if (tid == 0) {
          __builtin_amdgcn_fence(__ATOMIC_RELEASE, "");
          asm volatile("s_waitcnt vmcnt(0)" ::: "memory");
#pragma unroll
          for (int d = 1; d <= NR; ++d)
            __hip_atomic_fetch_add((unsigned*)(tabs.ws[(rank + d) & 7] + CTL_CNT) + ks, 1u,
                                   __ATOMIC_RELAXED, __HIP_MEMORY_SCOPE_SYSTEM);
        }
      }
    }
    return;
  }

  const int gb = blockIdx.x - NPUSH;
  const int xcd = gb & 7, idx = gb >> 3;
  const int mb = 4 * (xcd >> 1) + (idx & 3);
  const int nb = 8 * (xcd & 1) + (idx >> 2);
  const int wm = wv >> 1, wn = wv & 1;
  const char* xa = myws + stg + (size_t)mb * SLOT_A;
  const char* wb = myws + stg + XA_BYTES + (size_t)nb * SLOT_B;
  const unsigned* cnt = (const unsigned*)(myws + CTL_CNT);
  unsigned* tmo = (unsigned*)(myws + CTL_TMO);
  const unsigned target = CONTRIB * call;

  unsigned long long avail = 0ull;
  bool poisoned = false;
  auto ensure = [&](int ks) {
    if ((avail >> ks) & 1ull) return;
    const unsigned long long t0 = __builtin_amdgcn_s_memrealtime();
    for (;;) {
      const unsigned v = ld_sys(cnt + lane);
      avail = __ballot((int)(v - target) >= 0);
      if ((avail >> ks) & 1ull) break;
      if (__hip_atomic_load(tmo, __ATOMIC_RELAXED, __HIP_MEMORY_SCOPE_AGENT) == call ||
          __builtin_amdgcn_s_memrealtime() - t0 > DEADLINE_TICKS) {
        if (lane == 0) __hip_atomic_store(tmo, call, __ATOMIC_RELAXED, __HIP_MEMORY_SCOPE_AGENT);
        poisoned = true;
        avail = ~0ull;
        break;
      }
      __builtin_amdgcn_s_sleep(2);
    }
    __builtin_amdgcn_fence(__ATOMIC_ACQUIRE, "");
    asm volatile("s_waitcnt vmcnt(0)" ::: "memory");
  };
  auto slice_of = [&](int j) {
    const int i = j >> 1;
    return ((((rank + (i & 7)) & 7) * NU + (i >> 3)) << 1) | (j & 1);
  };
  auto issue = [&](int j) {
    const int sl = slice_of(j);
    char* slot = lds + (j % NSLOT) * SLOT;
    const char* ga = xa + (size_t)sl * XA_SLICE + wv * 2048 + lane * 16;
    const char* gbp = wb + (size_t)sl * WB_SLICE + wv * 1024 + lane * 16;
    __builtin_amdgcn_global_load_lds((const void*)ga, (__attribute__((address_space(3))) void*)(slot + wv * 2048), 16, 0, 0);
    __builtin_amdgcn_global_load_lds((const void*)(ga + 1024), (__attribute__((address_space(3))) void*)(slot + wv * 2048 + 1024), 16, 0, 0);
    __builtin_amdgcn_global_load_lds((const void*)gbp, (__attribute__((address_space(3))) void*)(slot + SLOT_A + wv * 1024), 16, 0, 0);
  };

  f32x4 acc[4][4];
#pragma unroll
  for (int i = 0; i < 4; ++i)
#pragma unroll
    for (int j = 0; j < 4; ++j) acc[i][j] = (f32x4){0.f, 0.f, 0.f, 0.f};

  if (wv == 0) ensure(slice_of(0) >> 1);
  __builtin_amdgcn_s_barrier();
  issue(0);
  issue(1);
  for (int j = 0; j < NSL; ++j) {
    if (wv == 0 && j + 2 < NSL && !(j & 1)) ensure(slice_of(j + 2) >> 1);
    if (j + 2 < NSL) asm volatile("s_waitcnt vmcnt(3)" ::: "memory");
    else asm volatile("s_waitcnt vmcnt(0)" ::: "memory");
    __builtin_amdgcn_s_barrier();
    if (j + 2 < NSL) issue(j + 2);
    const char* sa = lds + (j % NSLOT) * SLOT + wm * 4096 + lane * 16;
    const char* sb = lds + (j % NSLOT) * SLOT + SLOT_A + wn * 4096 + lane * 16;
    half8 a[4], b[4];
#pragma unroll
    for (int i = 0; i < 4; ++i) a[i] = *(const half8*)(sa + i * 1024);
#pragma unroll
    for (int i = 0; i < 4; ++i) b[i] = *(const half8*)(sb + i * 1024);
#pragma unroll
    for (int i = 0; i < 4; ++i)
#pragma unroll
      for (int jj = 0; jj < 4; ++jj)
        acc[i][jj] = __builtin_amdgcn_mfma_f32_16x16x32_f16(a[i], b[jj], acc[i][jj], 0, 0, 0);
  }
  if (wv == 0 && lane == 0) misc[1] = poisoned ? 1u : 0u;
  __syncthreads();
  const bool bad = misc[1] != 0u;

  {
    const int owner = mb >> 1;
    float* obase = tabs.out[owner] + (size_t)((mb & 1) * 256 + wm * 64) * NTOT +
                   (size_t)rank * NLOC + nb * 128 + wn * 64;
    char* reg = lds + wv * 9216;
#pragma unroll
    for (int half = 0; half < 2; ++half) {
#pragma unroll
      for (int i2 = 0; i2 < 2; ++i2)
#pragma unroll
        for (int jj = 0; jj < 4; ++jj)
#pragma unroll
          for (int r = 0; r < 4; ++r) {
            float v = silu(acc[half * 2 + i2][jj][r]);
            if (bad) v = __builtin_nanf("");
            *(float*)(reg + (i2 * 16 + (lane >> 4) * 4 + r) * 272 + (jj * 16 + (lane & 15)) * 4) = v;
          }
      asm volatile("s_waitcnt lgkmcnt(0)" ::: "memory");
#pragma unroll
      for (int s = 0; s < 8; ++s) {
        const int row = s * 4 + (lane >> 4);
        const f32x4 v = *(const f32x4*)(reg + row * 272 + (lane & 15) * 16);
        *(f32x4*)(obase + (size_t)(half * 32 + row) * NTOT + (lane & 15) * 4) = v;
      }
      asm volatile("s_waitcnt lgkmcnt(0)" ::: "memory");
    }
    asm volatile("s_waitcnt vmcnt(0)" ::: "memory");
    __syncthreads();
    if (tid == 0 && owner != rank) {
      __builtin_amdgcn_fence(__ATOMIC_RELEASE, "");
      asm volatile("s_waitcnt vmcnt(0)" ::: "memory");
      __hip_atomic_fetch_add((unsigned*)(tabs.ws[owner] + CTL_DONE), 1u, __ATOMIC_RELAXED,
                             __HIP_MEMORY_SCOPE_SYSTEM);
    }
  }

  if (gb == 0 && tid == 0) {
    const unsigned* done = (const unsigned*)(myws + CTL_DONE);
    const unsigned dt = DONE_CONTRIB * call;
    const unsigned long long t0 = __builtin_amdgcn_s_memrealtime();
    while ((int)(ld_sys(done) - dt) < 0) {
      if (__hip_atomic_load(tmo, __ATOMIC_RELAXED, __HIP_MEMORY_SCOPE_AGENT) == call ||
          __builtin_amdgcn_s_memrealtime() - t0 > DEADLINE_TICKS) {
        __hip_atomic_store(tmo, call, __ATOMIC_RELAXED, __HIP_MEMORY_SCOPE_AGENT);
        tabs.out[rank][0] = __builtin_nanf("");
        break;
      }
      __builtin_amdgcn_s_sleep(2);
    }
    __builtin_amdgcn_fence(__ATOMIC_ACQUIRE, "");
  }
}

extern "C" void kernel_launch(int rank, int nranks, ncclComm_t comm,
                              hipStream_t stream,
                              void* const* d_in, const int* in_sizes, int n_in,
                              void* d_out, int out_size,
                              const int* in_dtypes, int out_dtype,
                              void* const* d_out_all,
                              void* const* d_ws_all, size_t ws_bytes) {
  if (nranks != NR || (unsigned)rank >= (unsigned)NR || ws_bytes < STG_BASE + 2 * STG_BYTES) {
    fprintf(stderr, "unsupported configuration\n");
    return;
  }
  static bool init = false;
  if (!init) {
    hipError_t e = hipFuncSetAttribute((const void*)fused_gemm_rs,
                                       hipFuncAttributeMaxDynamicSharedMemorySize, LDS_BYTES);
    if (e != hipSuccess) fprintf(stderr, "hipFuncSetAttribute: %s\n", hipGetErrorString(e));
    init = true;
  }
  Tabs tabs;
  for (int p = 0; p < NR; ++p) {
    tabs.ws[p] = (char*)d_ws_all[p];
    tabs.out[p] = (float*)d_out_all[p];
  }
  hipLaunchKernelGGL(fused_gemm_rs, dim3(NWG), dim3(NTHR), LDS_BYTES, stream,
                     (const float*)d_in[0], (const float*)d_in[1], tabs, rank);
}
```

```cpp
#include <hip/hip_runtime.h>
#include <rccl/rccl.h>
#include <cstdio>
#include <cstdint>

typedef _Float16 half8 __attribute__((ext_vector_type(8)));
typedef float f32x4 __attribute__((ext_vector_type(4)));

namespace {

constexpr int NR = 8;
constexpr int MTOT = 4096;
constexpr int KC = 512;
constexpr int NTOT = 16384;
constexpr int NLOC = NTOT / NR;
constexpr int MLOC = MTOT / NR;
constexpr int NU = 8;
constexpr int NKS = NR * NU;
constexpr int NSL = NKS * 2;

constexpr size_t CTL_CNT = 0;
constexpr size_t CTL_DONE = 0x100;
constexpr size_t CTL_TICKET = 0x200;
constexpr size_t CTL_TMO = 0x300;
constexpr size_t CTL_QUEUE = 0x1000;
constexpr size_t CTL_RTICK = 0x400;
constexpr size_t STG_BASE = 1u << 20;
constexpr size_t XA_SLICE = 256u << 10;
constexpr size_t WB_SLICE = 128u << 10;
constexpr size_t XA_BYTES = XA_SLICE * NSL;
constexpr size_t WB_BYTES = WB_SLICE * NSL;
constexpr size_t STG_BYTES = XA_BYTES + WB_BYTES;

#define NPUSH_ 64
constexpr int NPUSH = NPUSH_;
constexpr int NGEMM = 256;
constexpr int NWG = NPUSH + NGEMM;
constexpr int NTHR = 512;

constexpr int W_ITEMS = 64, X_ITEMS = 32, R_ITEMS = W_ITEMS + X_ITEMS;
constexpr int ITEMS = R_ITEMS * NU;
constexpr unsigned CONTRIB = 8 + 32;
constexpr unsigned DONE_CONTRIB = 32 * (NR - 1);

constexpr unsigned long long DEADLINE_TICKS = 50000000ull;

constexpr int SLOT_A = 16384, SLOT_B = 8192, SLOT = SLOT_A + SLOT_B;
constexpr int NSLOT = 3;
constexpr int LDS_MISC = NSLOT * SLOT;
constexpr int LDS_BYTES = LDS_MISC + 64;

#define PROF(idx)
#define PROFW0(idx)

struct Tabs {
  char* ws[NR];
  float* out[NR];
};

__device__ __forceinline__ unsigned ld_sys(const unsigned* p) {
  return __hip_atomic_load(p, __ATOMIC_RELAXED, __HIP_MEMORY_SCOPE_SYSTEM);
}

typedef int int32x4 __attribute__((ext_vector_type(4)));
typedef unsigned uint32x4 __attribute__((ext_vector_type(4)));
__device__ __forceinline__ void st_wt(char* base, unsigned off, half8 v) {
  __amdgpu_buffer_rsrc_t r = __builtin_amdgcn_make_buffer_rsrc(base, (short)0, (int)0x7fffffff, 0x00020000);
  __builtin_amdgcn_raw_buffer_store_b128(__builtin_bit_cast(uint32x4, v), r, (int)off, 0, 17);
}

__device__ __forceinline__ half8 cvt8(const float4& a, const float4& b) {
  half8 h;
  h[0] = (_Float16)a.x; h[1] = (_Float16)a.y; h[2] = (_Float16)a.z; h[3] = (_Float16)a.w;
  h[4] = (_Float16)b.x; h[5] = (_Float16)b.y; h[6] = (_Float16)b.z; h[7] = (_Float16)b.w;
  return h;
}

__device__ __forceinline__ float silu(float y) {
  float c = fminf(fmaxf(y, -60.0f), 60.0f);
  return y / (1.0f + __expf(-c));
}

}

__global__ __launch_bounds__(NTHR, 4) void fused_gemm_rs(const float* __restrict__ x,
                                                         const float* __restrict__ w,
                                                         Tabs tabs, int rank) {
  extern __shared__ __attribute__((aligned(16))) char lds[];
  const int tid = threadIdx.x;
  const int lane = tid & 63;
  const int wv = tid >> 6;
  char* myws = tabs.ws[rank];
  volatile unsigned* misc = (volatile unsigned*)(lds + LDS_MISC);
  PROFW0(0);

  if (tid == 0) {
    unsigned t = __hip_atomic_fetch_add((unsigned*)(myws + CTL_TICKET), 1u, __ATOMIC_RELAXED,
                                        __HIP_MEMORY_SCOPE_AGENT);
    misc[0] = t / NWG + 1u;
    misc[1] = 0u;
  }
  __syncthreads();
  const unsigned call = misc[0];
  const size_t stg = STG_BASE + (size_t)(call & 1u) * STG_BYTES;

  if ((int)blockIdx.x < NPUSH) {
    unsigned* lcnt = (unsigned*)lds;
    if (tid < NU) lcnt[tid] = 0u;
    __syncthreads();
    char* xd0 = tabs.ws[(rank + 1 + (wv >> 1) * 2) & 7];
    char* xd1 = tabs.ws[(rank + 2 + (wv >> 1) * 2) & 7];
    auto signal = [&](int u) {
      if (lane == 0) {
        if (__hip_atomic_fetch_add(lcnt + u, 1u, __ATOMIC_RELAXED, __HIP_MEMORY_SCOPE_WORKGROUP) == 7u) {
          unsigned t = __hip_atomic_fetch_add((unsigned*)(myws + CTL_RTICK) + u * 32, 1u, __ATOMIC_RELAXED,
                                              __HIP_MEMORY_SCOPE_AGENT);
          if ((t + 1u) % NPUSH == 0u) {
#pragma unroll
            for (int d = 1; d <= NR; ++d)
              __hip_atomic_store((unsigned*)(tabs.ws[(rank + d) & 7] + CTL_CNT) + rank * NU + u, call,
                                 __ATOMIC_RELAXED, __HIP_MEMORY_SCOPE_SYSTEM);
          }
        }
      }
      PROF(1 + 8 * wv + u);
    };
    constexpr int SUB = 256 / NPUSH, NSTEP = NU * SUB;
    auto vb_of = [&](int st) { return (int)blockIdx.x + (st % SUB) * NPUSH; };
    auto wsrc_of = [&](int st) {
      const int vb = vb_of(st);
      return w + (size_t)(64 * (st / SUB) + 8 * wv) * NTOT + (NLOC * ((vb + rank) & 7) + 64 * (vb >> 3) + lane);
    };
    auto xsrc_of = [&](int st) {
      return x + (size_t)(16 * vb_of(st) + (lane & 15)) * KC + 64 * (st / SUB) + 32 * (wv & 1) + 8 * (lane >> 4);
    };
    float wvv[2][8];
    float4 a0[2], a1[2];
    {
      const float* ws0 = wsrc_of(0);
      const float* xs0 = xsrc_of(0);
#pragma unroll
      for (int j = 0; j < 8; ++j) wvv[0][j] = ws0[(size_t)j * NTOT];
      a0[0] = *(const float4*)(xs0);
      a1[0] = *(const float4*)(xs0 + 4);
    }
#pragma unroll 2
    for (int st = 0; st < NSTEP; ++st) {
      const int c = st & 1, n = c ^ 1;
      if (st + 1 < NSTEP) {
        const float* ws1 = wsrc_of(st + 1);
        const float* xs1 = xsrc_of(st + 1);
#pragma unroll
        for (int j = 0; j < 8; ++j) wvv[n][j] = ws1[(size_t)j * NTOT];
        a0[n] = *(const float4*)(xs1);
        a1[n] = *(const float4*)(xs1 + 4);
      }
      half8 hw, hx;
#pragma unroll
      for (int j = 0; j < 8; ++j) hw[j] = (_Float16)wvv[c][j];
      hx = cvt8(a0[c], a1[c]);
      const int u = st / SUB, vb = vb_of(st);
      const int ks = rank * NU + u;
      const size_t xoff = stg + (size_t)(ks * 2 + (wv & 1)) * XA_SLICE + (size_t)vb * 1024 + lane * 16;
      const size_t woff = stg + XA_BYTES + (size_t)(ks * 2 + (wv >> 2)) * WB_SLICE +
                          (size_t)((vb >> 3) * 4 + (lane >> 4)) * 1024 + ((wv & 3) * 16 + (lane & 15)) * 16;
      st_wt(tabs.ws[(vb + rank) & 7], (unsigned)woff, hw);
      st_wt(xd0, (unsigned)xoff, hx);
      st_wt(xd1, (unsigned)xoff, hx);
      if (st >= 2 && (st - 2) % SUB == SUB - 1) { asm volatile("" ::: "memory"); signal((st - 2) / SUB); }
    }
    if (SUB == 1) { asm volatile("s_waitcnt vmcnt(3)" ::: "memory"); signal(NU - 2); }
    asm volatile("s_waitcnt vmcnt(0)" ::: "memory");
    signal(NU - 1);
    return;
  }

  const int gb = blockIdx.x - NPUSH;
  const int xcd = gb & 7, idx = gb >> 3;
  const int mb = 4 * (xcd >> 1) + (idx & 3);
  const int nb = 8 * (xcd & 1) + (idx >> 2);
  const int wm = wv >> 1, wn = wv & 1;
  const char* xa = myws + stg + (size_t)mb * SLOT_A;
  const char* wb = myws + stg + XA_BYTES + (size_t)nb * SLOT_B;
  const unsigned* cnt = (const unsigned*)(myws + CTL_CNT);
  unsigned* tmo = (unsigned*)(myws + CTL_TMO);
  const unsigned target = call;

  unsigned long long remaining = ~0ull;
  const unsigned long long own = 0xffull << (rank * NU);
  bool poisoned = false;
  unsigned long long known = 0ull;
  unsigned vpend = 0;
  auto choose = [&]() -> int {
    known |= __ballot((int)(vpend - target) >= 0);
    vpend = ld_sys(cnt + lane);
    unsigned long long m = known & remaining & ~own;
    if (!m) m = known & remaining;
    if (!m) {
      const unsigned long long t0 = __builtin_amdgcn_s_memrealtime();
      for (;;) {
        known |= __ballot((int)(ld_sys(cnt + lane) - target) >= 0);
        m = known & remaining;
        if (m) break;
        if (poisoned || __hip_atomic_load(tmo, __ATOMIC_RELAXED, __HIP_MEMORY_SCOPE_AGENT) == call ||
            __builtin_amdgcn_s_memrealtime() - t0 > DEADLINE_TICKS) {
          if (lane == 0) __hip_atomic_store(tmo, call, __ATOMIC_RELAXED, __HIP_MEMORY_SCOPE_AGENT);
          poisoned = true;
          m = remaining;
          break;
        }
        __builtin_amdgcn_s_sleep(1);
      }
      asm volatile("" ::: "memory");
    }
    const int ks = __builtin_ctzll(m);
    remaining &= ~(1ull << ks);
    return ks;
  };
  auto issue = [&](int j, int unit) {
    const int sl = unit * 2 + (j & 1);
    char* slot = lds + (j % NSLOT) * SLOT;
    const char* ga = xa + (size_t)sl * XA_SLICE + wv * 2048 + lane * 16;
    const char* gbp = wb + (size_t)sl * WB_SLICE + wv * 1024 + lane * 16;
    __builtin_amdgcn_global_load_lds((const void*)ga, (__attribute__((address_space(3))) void*)(slot + wv * 2048), 16, 0, 0);
    __builtin_amdgcn_global_load_lds((const void*)(ga + 1024), (__attribute__((address_space(3))) void*)(slot + wv * 2048 + 1024), 16, 0, 0);
    __builtin_amdgcn_global_load_lds((const void*)gbp, (__attribute__((address_space(3))) void*)(slot + SLOT_A + wv * 1024), 16, 0, 0);
  };

  f32x4 acc[4][4];
#pragma unroll
  for (int i = 0; i < 4; ++i)
#pragma unroll
    for (int j = 0; j < 4; ++j) acc[i][j] = (f32x4){0.f, 0.f, 0.f, 0.f};

  if (wv == 0) { const int u0 = choose(); if (lane == 0) misc[4] = (unsigned)u0; }
  __syncthreads();
  int unit = (int)misc[4];
  issue(0, unit);
  issue(1, unit);
  for (int j = 0; j < NSL; ++j) {
    const bool pick = !(j & 1) && j + 2 < NSL;
    if (wv == 0 && pick) {
      const int un = choose();
      if (lane == 0) misc[4 + (((j >> 1) + 1) & 3)] = (unsigned)un;
      PROF(1 + (j >> 1));
    }
    if (j + 2 >= NSL) asm volatile("s_waitcnt vmcnt(0) lgkmcnt(0)" ::: "memory");
    else if (wv == 0 && (j & 1)) asm volatile("s_waitcnt vmcnt(4) lgkmcnt(0)" ::: "memory");
    else asm volatile("s_waitcnt vmcnt(3) lgkmcnt(0)" ::: "memory");
    __builtin_amdgcn_s_barrier();
    if (pick) unit = (int)misc[4 + (((j >> 1) + 1) & 3)];
    if (j + 2 < NSL) issue(j + 2, unit);
    const char* sa = lds + (j % NSLOT) * SLOT + wm * 4096 + lane * 16;
    const char* sb = lds + (j % NSLOT) * SLOT + SLOT_A + wn * 4096 + lane * 16;
    half8 a[4], b[4];
#pragma unroll
    for (int i = 0; i < 4; ++i) a[i] = *(const half8*)(sa + i * 1024);
#pragma unroll
    for (int i = 0; i < 4; ++i) b[i] = *(const half8*)(sb + i * 1024);
#pragma unroll
    for (int i = 0; i < 4; ++i)
#pragma unroll
      for (int jj = 0; jj < 4; ++jj)
        acc[i][jj] = __builtin_amdgcn_mfma_f32_16x16x32_f16(a[i], b[jj], acc[i][jj], 0, 0, 0);
  }
  PROFW0(65);
  if (wv == 0 && lane == 0) misc[1] = poisoned ? 1u : 0u;
  __syncthreads();
  const bool bad = misc[1] != 0u;

  {
    const int owner = mb >> 1;
    float* obase = tabs.out[owner] + (size_t)((mb & 1) * 256 + wm * 64) * NTOT +
                   (size_t)rank * NLOC + nb * 128 + wn * 64;
    char* reg = lds + wv * 9216;
#pragma unroll
    for (int half = 0; half < 2; ++half) {
#pragma unroll
      for (int i2 = 0; i2 < 2; ++i2)
#pragma unroll
        for (int jj = 0; jj < 4; ++jj)
#pragma unroll
          for (int r = 0; r < 4; ++r) {
            float v = silu(acc[half * 2 + i2][jj][r]);
            if (bad) v = __builtin_nanf("");
            *(float*)(reg + (i2 * 16 + (lane >> 4) * 4 + r) * 272 + (jj * 16 + (lane & 15)) * 4) = v;
          }
      asm volatile("s_waitcnt lgkmcnt(0)" ::: "memory");
#pragma unroll
      for (int s = 0; s < 8; ++s) {
        const int row = s * 4 + (lane >> 4);
        const f32x4 v = *(const f32x4*)(reg + row * 272 + (lane & 15) * 16);
        *(f32x4*)(obase + (size_t)(half * 32 + row) * NTOT + (lane & 15) * 4) = v;
      }
      asm volatile("s_waitcnt lgkmcnt(0)" ::: "memory");
    }
    asm volatile("s_waitcnt vmcnt(0)" ::: "memory");
    __syncthreads();
    if (tid == 0 && owner != rank) {
      __builtin_amdgcn_fence(__ATOMIC_RELEASE, "");
      asm volatile("s_waitcnt vmcnt(0)" ::: "memory");
      __hip_atomic_fetch_add((unsigned*)(tabs.ws[owner] + CTL_DONE), 1u, __ATOMIC_RELAXED,
                             __HIP_MEMORY_SCOPE_SYSTEM);
    }
  }

  PROFW0(66);
  if (gb == 0 && tid == 0) {
    const unsigned* done = (const unsigned*)(myws + CTL_DONE);
    const unsigned dt = DONE_CONTRIB * call;
    const unsigned long long t0 = __builtin_amdgcn_s_memrealtime();
    while ((int)(ld_sys(done) - dt) < 0) {
      if (__hip_atomic_load(tmo, __ATOMIC_RELAXED, __HIP_MEMORY_SCOPE_AGENT) == call ||
          __builtin_amdgcn_s_memrealtime() - t0 > DEADLINE_TICKS) {
        __hip_atomic_store(tmo, call, __ATOMIC_RELAXED, __HIP_MEMORY_SCOPE_AGENT);
        tabs.out[rank][0] = __builtin_nanf("");
        break;
      }
      __builtin_amdgcn_s_sleep(2);
    }
    __builtin_amdgcn_fence(__ATOMIC_ACQUIRE, "");
    PROF(67);
  }
}

extern "C" void kernel_launch(int rank, int nranks, ncclComm_t comm,
                              hipStream_t stream,
                              void* const* d_in, const int* in_sizes, int n_in,
                              void* d_out, int out_size,
                              const int* in_dtypes, int out_dtype,
                              void* const* d_out_all,
                              void* const* d_ws_all, size_t ws_bytes) {
  if (nranks != NR || (unsigned)rank >= (unsigned)NR || ws_bytes < STG_BASE + 2 * STG_BYTES) {
    fprintf(stderr, "unsupported configuration\n");
    return;
  }
  static bool init = false;
  if (!init) {
    hipError_t e = hipFuncSetAttribute((const void*)fused_gemm_rs,
                                       hipFuncAttributeMaxDynamicSharedMemorySize, LDS_BYTES);
    if (e != hipSuccess) fprintf(stderr, "hipFuncSetAttribute: %s\n", hipGetErrorString(e));
    init = true;
  }
  Tabs tabs;
  for (int p = 0; p < NR; ++p) {
    tabs.ws[p] = (char*)d_ws_all[p];
    tabs.out[p] = (float*)d_out_all[p];
  }
  hipLaunchKernelGGL(fused_gemm_rs, dim3(NWG), dim3(NTHR), LDS_BYTES, stream,
                     (const float*)d_in[0], (const float*)d_in[1], tabs, rank);
}
```
